```python
import numpy as np

kernel_name = "gpu_gemm_ar_m4096_k4096_n4096_e5m2e5m2_dequant"


ABSMAX_RTOL = 0.05

COMPUTE_DTYPE = "float32"
EPILOGUE = "dequant"
IN_QUANT = None
OUT_QUANT = None
IN_FP8 = "e5m2e5m2"
IN_DTYPES = {"x": "e5m2", "w_mat": "e5m2",
             "scale_x": "float32", "scale_w": "float32"}
WORLD_SIZE = 8

def _snap_fp8(x, mant_bits, max_abs, min_normal):
    a = np.abs(x.astype(np.float64))
    _m, _e = np.frexp(a)
    q = np.where(a >= min_normal, np.ldexp(1.0, _e - 1 - mant_bits),
                 min_normal * 2.0 ** -mant_bits)
    snapped = np.minimum(np.round(a / q) * q, max_abs)
    return (np.sign(x) * snapped).astype(np.float32)

def setup_inputs():

    x = _snap_fp8(np.random.randn(4096, 4096).astype(np.float32), 2, 57344.0, 2.0 ** -14)
    w_mat = _snap_fp8(np.random.randn(4096, 4096).astype(np.float32), 2, 57344.0, 2.0 ** -14)
    scale_x = (0.02 * (1.0 + np.random.rand(1))).astype(np.float32)
    scale_w = (0.02 * (1.0 + np.random.rand(1))).astype(np.float32)
    return {"x": x, "w_mat": w_mat, "scale_x": scale_x, "scale_w": scale_w}

def reference(x, w_mat, scale_x, scale_w):
    acc = (x @ w_mat).astype(np.float32)
    y = (acc * (scale_x * scale_w)).astype(np.float32)
    return y
```

```cpp
#include <hip/hip_runtime.h>
#include <rccl/rccl.h>
#include <cstdio>
#include <type_traits>


#define HIP_CHECK(expr) do { hipError_t _e = (expr); if (_e != hipSuccess) { \
    fprintf(stderr, "HIP error %s at %s:%d\n", hipGetErrorString(_e), __FILE__, __LINE__); \
    return; } } while (0)

typedef __attribute__((ext_vector_type(8))) int i32x8;
typedef __attribute__((ext_vector_type(4))) int i32x4;
typedef __attribute__((ext_vector_type(4))) unsigned u32x4;
typedef __attribute__((ext_vector_type(4))) float f32x4;

constexpr int NR = 8;
constexpr int MD = 4096, ND = 4096;
constexpr int KS = 512;
constexpr int NSTEP = 32;
constexpr int NPUSH = 16;
constexpr int TM = 288, TMS = 272, TN = 256;
constexpr int NTM = 15, NTN = 16;
constexpr int NGEMM = NTM * NTN;
constexpr int GRID = NPUSH + NGEMM;

constexpr size_t PIECE = 4096ull * 16;
constexpr unsigned S_ST = 2 * PIECE, B_OFF = PIECE, R_ST = NSTEP * S_ST;
constexpr size_t PARB = (size_t)NR * R_ST;
constexpr size_t FLAG_OFF = 2 * PARB;
constexpr size_t CNT_OFF = FLAG_OFF + (size_t)NR * NPUSH * 128;
constexpr size_t TMO_OFF = CNT_OFF + (size_t)GRID * 128;
constexpr size_t PROG_OFF = TMO_OFF + 128;
constexpr size_t WS_NEED = PROG_OFF + 128;

constexpr int A_ST = TM * 16, B_ST = TN * 16;
constexpr int A_BYTES = NR * A_ST;
constexpr int STAGE = A_BYTES + NR * B_ST;
constexpr int LDS_BYTES = 2 * STAGE;
constexpr int PA_ST = 4096 + 32;
constexpr int PB_OFF = 16 * PA_ST;
constexpr int PCTL = PB_OFF + 16 * 4096;
static_assert(PCTL + 64 <= LDS_BYTES, "pusher LDS");

constexpr unsigned long long DEADLINE_TICKS = 50000000ull;

struct Params {
  const unsigned char* x;
  const unsigned char* w;
  const float* sx;
  const float* sw;
  float* out;
  int rank;
  int pad;
  char* wsr[NR];
};

#define WAITV(n) asm volatile("s_waitcnt vmcnt(" #n ")" ::: "memory")
#define WAITL(n) asm volatile("s_waitcnt lgkmcnt(" #n ")" ::: "memory")
#define SCHED() __builtin_amdgcn_sched_barrier(0)
#define MFMAI(a, b, c) asm volatile("v_mfma_f32_16x16x128_f8f6f4 %0, %1, %2, %0 cbsz:1 blgp:1" : "+v"(c) : "v"(a), "v"(b))
#define LDSP(p) ((__attribute__((address_space(3))) void*)(p))
#define BLDS(rs, l, voff, soff, imm) __builtin_amdgcn_raw_ptr_buffer_load_lds(rs, LDSP(l), 16, voff, soff, imm, 17)
#define IC(x) std::integral_constant<int, (x)>{}

constexpr int WIN0 = 4, WIN1 = 3, WIN2 = 2;
constexpr int LVL_HI = 8;
constexpr int SKEW_S = 80;
constexpr int SKEW_RAMP = 16;
__host__ __device__ constexpr int pushA_of(int i) { return i < 8 ? 2 * i : 2 * (15 - i) + 1; }
__host__ __device__ constexpr int blkA_of(int q) { return (q & 1) ? 15 - (q >> 1) : (q >> 1); }
__host__ __device__ constexpr int pushB_of(int j) { return (j & 3) * 4 + (j >> 2); }
__host__ __device__ constexpr int blkB_of(int q) { return (q & 3) * 4 + (q >> 2); }
__host__ __device__ constexpr int skew_raw(int q) { return SKEW_S - (SKEW_S * q * q) / 256; }
__host__ __device__ constexpr int skew_mean() { int t = 0; for (int i = 0; i < 16; ++i) t += skew_raw(i); return t / 16; }
__host__ __device__ constexpr int skew16(int q) { return skew_raw(q) - skew_mean(); }
__host__ __device__ inline void tile_of(int b, int& tm, int& tn) {
  const int idx = (b & 7) * 30 + (b >> 3);
  const int band = idx / 60, r = idx % 60;
  tm = r >> 2; tn = band * 4 + (r & 3);
}
constexpr int younger(int j, int W) {
  const int t = j - W;
  if (t >= 0) return (j - 1 - t) + ((t < 4 && j > 4) ? 1 : 0);
  const int tp = 8 + t;
  return (7 - tp) + (tp < 4 ? 1 : 0) + 1 + j + (j > 4 ? 1 : 0);
}
#define WAITW(j) asm volatile("s_waitcnt vmcnt(%1)\n\ts_cmp_lt_i32 %0, 1\n\ts_cbranch_scc1 .Lw%=\n\ts_waitcnt vmcnt(%2)\n\t" \
    "s_cmp_lt_i32 %0, 2\n\ts_cbranch_scc1 .Lw%=\n\ts_waitcnt vmcnt(%3)\n.Lw%=:" \
    :: "s"(lvl), "i"(younger(j, WIN0)), "i"(younger(j, WIN1)), "i"(younger(j, WIN2)) : "memory", "scc")

__device__ __forceinline__ i32x8 rdfrag(const char* p, int st) {
  i32x4 lo = *(const i32x4*)(p);
  i32x4 hi = *(const i32x4*)(p + st);
  return __builtin_shufflevector(lo, hi, 0, 1, 2, 3, 4, 5, 6, 7);
}

__device__ __forceinline__ void tr16x4(const unsigned (&r)[16], u32x4 (&c)[4]) {
#pragma unroll
  for (int g = 0; g < 4; ++g) {
    const unsigned t0 = __builtin_amdgcn_perm(r[4 * g + 1], r[4 * g], 0x05010400u);
    const unsigned t1 = __builtin_amdgcn_perm(r[4 * g + 1], r[4 * g], 0x07030602u);
    const unsigned t2 = __builtin_amdgcn_perm(r[4 * g + 3], r[4 * g + 2], 0x05010400u);
    const unsigned t3 = __builtin_amdgcn_perm(r[4 * g + 3], r[4 * g + 2], 0x07030602u);
    c[0][g] = __builtin_amdgcn_perm(t2, t0, 0x05040100u);
    c[1][g] = __builtin_amdgcn_perm(t2, t0, 0x07060302u);
    c[2][g] = __builtin_amdgcn_perm(t3, t1, 0x05040100u);
    c[3][g] = __builtin_amdgcn_perm(t3, t1, 0x07060302u);
  }
}

__global__ void __launch_bounds__(512, 2) gemm_ag_kernel(Params P) {
  extern __shared__ __attribute__((aligned(16))) char smem[];
  const int wid = __builtin_amdgcn_readfirstlane(threadIdx.x >> 6);
  const int lane = threadIdx.x & 63;
  const int rank = P.rank;
  char* myws = P.wsr[0];
  char* dws = P.wsr[wid & 7];
  const unsigned char* px = P.x;
  const unsigned char* pw = P.w;
  asm volatile("" :: "s"(myws), "s"(dws), "s"(px), "s"(pw));
  unsigned* cntp = (unsigned*)(myws + CNT_OFF + (size_t)blockIdx.x * 128 + wid * 16);
  unsigned* tmo = (unsigned*)(myws + TMO_OFF);

  if (__builtin_expect(blockIdx.x < NPUSH, 1)) {
    const int q = blockIdx.x;
    const int qa = blkA_of(q), qb = blkB_of(q);
    unsigned call = __hip_atomic_load(cntp, __ATOMIC_RELAXED, __HIP_MEMORY_SCOPE_AGENT) + 1;
    if (wid < 4) {
      const unsigned char* wsrc = pw + (size_t)(16 * wid) * ND + 256 * qb + 4 * lane;
      unsigned lb[16];
      u32x4 sc[4];
#pragma unroll
      for (int kk = 0; kk < 16; ++kk) lb[kk] = *(const unsigned*)(wsrc + (size_t)kk * ND);
      tr16x4(lb, sc);
#pragma unroll
      for (int i = 0; i < 4; ++i)
        *(u32x4*)(smem + PB_OFF + wid * 4096 + (64 * i + lane) * 16) = sc[i];
    } else {
      const int r0 = 64 * (wid - 4) + (lane >> 2);
      const unsigned char* xs = px + (size_t)(256 * qa + r0) * KS + (lane & 3) * 16;
      u32x4 la[4];
#pragma unroll
      for (int i = 0; i < 4; ++i) la[i] = *(const u32x4*)(xs + (size_t)(16 * i) * KS);
#pragma unroll
      for (int i = 0; i < 4; ++i) *(u32x4*)(smem + (lane & 3) * PA_ST + (r0 + 16 * i) * 16) = la[i];
    }
    call = __builtin_amdgcn_readfirstlane(call);
    const unsigned base = call * 64u;
    __amdgpu_buffer_rsrc_t rs = __builtin_amdgcn_make_buffer_rsrc(dws + (size_t)(call & 1) * PARB, 0, (unsigned)PARB, 0x00020000);
    const unsigned offA = (unsigned)rank * R_ST + (256 * qa + lane) * 16;
    const unsigned offB = (unsigned)rank * R_ST + B_OFF + (256 * qb + lane) * 16;
    volatile __attribute__((address_space(3))) int* ctl = (volatile __attribute__((address_space(3))) int*)LDSP(smem + PCTL);
    if (threadIdx.x < 9) ctl[threadIdx.x] = (threadIdx.x == 0) ? 1 : 0;
    WAITL(0);
    __builtin_amdgcn_s_barrier();
    auto lds_wait = [&](int idx, int need) {
      while (__builtin_amdgcn_readfirstlane(ctl[idx]) < need) __builtin_amdgcn_s_sleep(1);
      asm volatile("" ::: "memory");
    };
    if (wid != 0) {
      __amdgpu_buffer_rsrc_t rf = __builtin_amdgcn_make_buffer_rsrc(dws + FLAG_OFF + (size_t)(rank * NPUSH + q) * 128, 0, 4, 0x00020000);
      __amdgpu_buffer_rsrc_t rp = __builtin_amdgcn_make_buffer_rsrc(myws + PROG_OFF, 0, 4, 0x00020000);
      const unsigned foff = lane == 0 ? 0u : 64u;
      const unsigned pbase = (call - 1) * (unsigned)(NPUSH * 7 * NSTEP);
      const int skew = skew16(q);
      int lvl = 0, staged = 1;
#pragma unroll 1
      for (int s = 0; s < NSTEP; ++s) {
        if ((s >> 2) >= staged) { lds_wait(0, (s >> 2) + 1); staged = __builtin_amdgcn_readfirstlane(ctl[0]); }
        u32x4 sa[4], sb[4];
#pragma unroll
        for (int i = 0; i < 4; ++i) sa[i] = *(const u32x4*)(smem + (s & 15) * PA_ST + i * 1024 + lane * 16);
#pragma unroll
        for (int i = 0; i < 4; ++i) sb[i] = *(const u32x4*)(smem + PB_OFF + (s & 15) * 4096 + i * 1024 + lane * 16);
        asm volatile("" ::: "memory");
        const unsigned so = (unsigned)s * S_ST;
        int prog = __builtin_amdgcn_raw_ptr_buffer_atomic_add_i32(1, rp, foff, 0, 16);
        WAITW(0);
        __builtin_amdgcn_raw_buffer_store_b128(sa[0], rs, offA + so, 0, 17);
        WAITW(1);
        __builtin_amdgcn_raw_buffer_store_b128(sa[1], rs, offA + so + 1024, 0, 17);
        WAITW(2);
        __builtin_amdgcn_raw_buffer_store_b128(sa[2], rs, offA + so + 2048, 0, 17);
        WAITW(3);
        __builtin_amdgcn_raw_buffer_store_b128(sa[3], rs, offA + so + 3072, 0, 17);
        WAITW(4);
        __builtin_amdgcn_raw_buffer_store_b32(base + s, rf, ((s & 1) == 0 || s > 22) ? foff : 64u, 0, 17);
        asm volatile("" ::: "memory");
        __builtin_amdgcn_raw_buffer_store_b128(sb[0], rs, offB + so, 0, 17);
        WAITW(5);
        asm volatile("" : "+v"(prog));
        {
          const int cnt = (int)((unsigned)__builtin_amdgcn_readfirstlane(prog) - pbase);
          const int d = 112 * s + 56 - cnt - 7 * ((skew * min(s, SKEW_RAMP)) / SKEW_RAMP);
          const bool endgame = 16 * s + skew16(0) - skew >= 16 * (NSTEP - 1) - 8;
          lvl = __builtin_amdgcn_readfirstlane(endgame ? 0 : (d > 7 * LVL_HI) ? 2 : (d > 0) ? 1 : 0);
        }
        __builtin_amdgcn_raw_buffer_store_b128(sb[1], rs, offB + so + 1024, 0, 17);
        WAITW(6);
        __builtin_amdgcn_raw_buffer_store_b128(sb[2], rs, offB + so + 2048, 0, 17);
        WAITW(7);
        __builtin_amdgcn_raw_buffer_store_b128(sb[3], rs, offB + so + 3072, 0, 17);
        asm volatile("" ::: "memory");
        if ((s & 3) == 3 && lane == 0)
          __hip_atomic_fetch_add((__attribute__((address_space(3))) int*)(ctl + 1 + (s >> 2)), 1, __ATOMIC_RELAXED, __HIP_MEMORY_SCOPE_WORKGROUP);
      }
      WAITV(0);
      __builtin_amdgcn_raw_buffer_store_b32(base + NSTEP, rf, foff, 0, 17);
    } else {
      {
        const unsigned* of = (const unsigned*)(myws + FLAG_OFF + (size_t)(rank * NPUSH + q) * 128);
        const unsigned long long t0 = __builtin_amdgcn_s_memrealtime();
        while ((int)(__hip_atomic_load(of, __ATOMIC_RELAXED, __HIP_MEMORY_SCOPE_AGENT) - base) < 0) {
          if (__hip_atomic_load(tmo, __ATOMIC_RELAXED, __HIP_MEMORY_SCOPE_AGENT) == call ||
              __builtin_amdgcn_s_memrealtime() - t0 > DEADLINE_TICKS) {
            __hip_atomic_store(tmo, call, __ATOMIC_RELAXED, __HIP_MEMORY_SCOPE_AGENT);
            break;
          }
          __builtin_amdgcn_s_sleep(8);
        }
      }
#pragma unroll 1
      for (int c = 1; c < 8; ++c) {
        if (c >= 4) lds_wait(1 + c - 4, 7);
#pragma unroll 1
        for (int j = 0; j < 4; ++j) {
          const int s = 4 * c + j;
          const unsigned so = (unsigned)s * S_ST;
          char* pa = smem + (s & 15) * PA_ST;
          char* pb = smem + PB_OFF + (s & 15) * 4096;
#define LW() do { if (c > 1) WAITV(3); } while (0)
          BLDS(rs, pa, offA, so, 0); BLDS(rs, pa, offA, so, 1024); LW();
          BLDS(rs, pa, offA, so, 2048); BLDS(rs, pa, offA, so, 3072); LW();
          BLDS(rs, pb, offB, so, 0); BLDS(rs, pb, offB, so, 1024); LW();
          BLDS(rs, pb, offB, so, 2048); BLDS(rs, pb, offB, so, 3072); LW();
        }
        WAITV(0);
        ctl[0] = c + 1;
      }
    }
    if (lane == 0) __hip_atomic_store(cntp, call, __ATOMIC_RELAXED, __HIP_MEMORY_SCOPE_AGENT);
    WAITV(0);
    return;
  }

  int tm, tn;
  tile_of(blockIdx.x - NPUSH, tm, tn);
  unsigned call = __hip_atomic_load(cntp, __ATOMIC_RELAXED, __HIP_MEMORY_SCOPE_AGENT) + 1;
  call = __builtin_amdgcn_readfirstlane(call);
  const float scale = P.sx[0] * P.sw[0];
  if (lane == 0) __hip_atomic_store(cntp, call, __ATOMIC_RELAXED, __HIP_MEMORY_SCOPE_AGENT);
  const unsigned base = call * 64u;
  __amdgpu_buffer_rsrc_t rs = __builtin_amdgcn_make_buffer_rsrc(myws + (size_t)(call & 1) * PARB, 0, (unsigned)PARB, 0x00020000);
  if (blockIdx.x - NPUSH < 64) {
    const int q = (blockIdx.x - NPUSH) & 15, u = (blockIdx.x - NPUSH) >> 4;
    const int qa = blkA_of(q), qb = blkB_of(q);
    const unsigned char* xs = px + (size_t)(256 * qa + 32 * wid + (lane >> 3)) * KS + (lane & 7) * 16 + 128 * u;
    const unsigned char* wsrc = pw + (size_t)(128 * u + 16 * wid) * ND + 256 * qb + 4 * lane;
    u32x4 la[4], sc[4];
    unsigned lb[16];
#pragma unroll
    for (int i = 0; i < 4; ++i) la[i] = *(const u32x4*)(xs + (size_t)(8 * i) * KS);
#pragma unroll
    for (int kk = 0; kk < 16; ++kk) lb[kk] = *(const unsigned*)(wsrc + (size_t)kk * ND);
    const unsigned oa = (unsigned)rank * R_ST + (unsigned)(8 * u + (lane & 7)) * S_ST + (256 * qa + 32 * wid + (lane >> 3)) * 16;
    const unsigned ob = (unsigned)rank * R_ST + B_OFF + (unsigned)(8 * u + wid) * S_ST + (256 * qb + lane) * 16;
#pragma unroll
    for (int i = 0; i < 4; ++i) __builtin_amdgcn_raw_buffer_store_b128(la[i], rs, oa + i * 128, 0, 17);
    tr16x4(lb, sc);
#pragma unroll
    for (int i = 0; i < 4; ++i) __builtin_amdgcn_raw_buffer_store_b128(sc[i], rs, ob + i * 1024, 0, 17);
    WAITV(0);
    __builtin_amdgcn_s_barrier();
    if (threadIdx.x == 0)
      __hip_atomic_fetch_add((unsigned*)(myws + FLAG_OFF + (size_t)(rank * NPUSH + q) * 128), 16u, __ATOMIC_RELAXED, __HIP_MEMORY_SCOPE_AGENT);
  }
  const unsigned vA = (unsigned)wid * R_ST + (TMS * tm + lane) * 16;
  const unsigned vB = (unsigned)wid * R_ST + B_OFF + (TN * tn + lane) * 16;
  const unsigned* fl;
  {
    const int qq = (lane == 0) ? pushA_of(tm) : (lane == 1) ? pushA_of(tm + 1) : pushB_of(tn);
    fl = (lane < 3) ? (const unsigned*)(myws + FLAG_OFF + (size_t)(wid * NPUSH + qq) * 128) : tmo;
  }
  int ready = 0;
  unsigned pv;
  asm volatile("v_mov_b32 v255, %0" :: "v"(base - 1) : "v255");
#define POLL_ISSUE() asm volatile("global_load_dword v255, %0, off sc0 sc1" :: "v"(fl) : "memory", "v255")
#define POLL_WAIT() asm volatile("s_waitcnt vmcnt(0)\n\tv_mov_b32 %0, v255\n\ts_nop 1" : "=v"(pv) :: "memory")
  auto poll_use = [&]() {
    const int a = (int)((unsigned)__builtin_amdgcn_readlane(pv, 0) - base);
    const int b = (int)((unsigned)__builtin_amdgcn_readlane(pv, 1) - base);
    const int c = (int)((unsigned)__builtin_amdgcn_readlane(pv, 2) - base);
    int m = min(a, b);
    asm volatile("" : "+s"(m));
    m = min(m, c);
    asm volatile("" : "+s"(m));
    if (wid == rank && m >= 0) m = NSTEP;
    ready = max(ready, m);
  };
  auto wait_ready = [&](int need) {
    if (ready >= need) return;
    const unsigned long long t0 = __builtin_amdgcn_s_memrealtime();
    while (true) {
      POLL_ISSUE();
      POLL_WAIT();
      poll_use();
      if (ready >= need) break;
      if ((unsigned)__builtin_amdgcn_readlane(pv, 3) == call || __builtin_amdgcn_s_memrealtime() - t0 > DEADLINE_TICKS) {
        __hip_atomic_store(tmo, call, __ATOMIC_RELAXED, __HIP_MEMORY_SCOPE_AGENT);
        ready = NSTEP;
        break;
      }
      __builtin_amdgcn_s_sleep(1);
    }
    __builtin_amdgcn_s_waitcnt(0xC07F);
  };
  auto ld1 = [&](int s, auto ic) {
    constexpr int i = decltype(ic)::value;
    char* sA = smem + (s & 1) * STAGE + wid * A_ST;
    char* sB = smem + (s & 1) * STAGE + A_BYTES + wid * B_ST;
    const unsigned so = (unsigned)s * S_ST;
    if constexpr (i < 4) BLDS(rs, sB, vB, so, i * 1024);
    else if constexpr (i < 8) BLDS(rs, sA, vA, so, (i - 4) * 1024);
    else if constexpr (i == 8) { if (lane < 32) BLDS(rs, sA + 2048, vA + 2048, so, 2048); }
  };
#define LD1(s, i) ld1(s, IC(i))
  auto stage_load = [&](int s) {
    LD1(s, 0); LD1(s, 1); LD1(s, 2); LD1(s, 3); LD1(s, 4); LD1(s, 5); LD1(s, 6); LD1(s, 7); LD1(s, 8);
  };

  const int wm = wid >> 2, wn = wid & 3;
  const int fj = lane & 15, fg = lane >> 4;
  f32x4 acc[9][4];
#pragma unroll
  for (int m = 0; m < 9; ++m)
#pragma unroll
    for (int n = 0; n < 4; ++n) acc[m][n] = (f32x4){0.f, 0.f, 0.f, 0.f};
  const int aoff = (2 * fg) * A_ST + (wm * 144 + fj) * 16;
  const int boff = A_BYTES + (2 * fg) * B_ST + (wn * 16 + fj) * 16;

  i32x8 bf[4], af[3];
  auto mstep = [&](int s, const char* sb, auto mc, auto lc) {
    constexpr int m = decltype(mc)::value;
    constexpr int L0 = decltype(lc)::value;
    if constexpr (m + 2 < 9) af[(m + 2) % 3] = rdfrag(sb + aoff + (m + 2) * 256, A_ST);
    SCHED();
    MFMAI(af[m % 3], bf[0], acc[m][0]);
    MFMAI(af[m % 3], bf[1], acc[m][1]);
    SCHED();
    if constexpr (L0 >= 0) LD1(s + 1, L0);
    SCHED();
    MFMAI(af[m % 3], bf[2], acc[m][2]);
    MFMAI(af[m % 3], bf[3], acc[m][3]);
    SCHED();
    if constexpr (L0 >= 0) { LD1(s + 1, L0 + 1); if constexpr (L0 == 6) LD1(s + 1, 8); }
    SCHED();
  };
#define MS(m, l) mstep(s, sb, IC(m), IC(l))
  auto frags0 = [&](const char* sb) {
    bf[0] = rdfrag(sb + boff, B_ST);
    af[0] = rdfrag(sb + aoff, A_ST);
    SCHED();
    bf[1] = rdfrag(sb + boff + 1024, B_ST);
    SCHED();
    bf[2] = rdfrag(sb + boff + 2048, B_ST);
    SCHED();
    bf[3] = rdfrag(sb + boff + 3072, B_ST);
    SCHED();
    af[1] = rdfrag(sb + aoff + 256, A_ST);
    SCHED();
  };
  const bool full9 = (wm == 0) || (tm == NTM - 1);

  wait_ready(1);
  stage_load(0);
  POLL_WAIT();
#pragma unroll 1
  for (int s = 0; s < NSTEP - 1; ++s) {
    const bool ld1st = ready >= s + 2;
    const char* sb = smem + (s & 1) * STAGE;
    __builtin_amdgcn_s_waitcnt(0xC07F);
    SCHED();
    __builtin_amdgcn_s_barrier();
    SCHED();
    frags0(sb);
    int fast = __builtin_amdgcn_readfirstlane(ld1st ? 1 : 0), slow = fast ^ 1;
    asm volatile("" : "+s"(fast), "+s"(slow));
    bool issued = ld1st;
    if (fast) {
      MS(0, 0); MS(1, 2); MS(2, 4); MS(3, 6); MS(4, -1); MS(5, -1);
    }
    SCHED();
    if (slow) {
      POLL_ISSUE();
      SCHED();
      MS(0, -1); MS(1, -1); MS(2, -1);
      POLL_WAIT();
      poll_use();
      if (ready >= s + 2) { stage_load(s + 1); issued = true; }
      else POLL_ISSUE();
      SCHED();
      MS(3, -1); MS(4, -1); MS(5, -1);
      if (!issued) {
        POLL_WAIT();
        poll_use();
        if (ready >= s + 2) { stage_load(s + 1); issued = true; }
      }
    }
    if (ready < min(s + 3, NSTEP)) POLL_ISSUE();
    SCHED();
    MS(6, -1); MS(7, -1);
    if (full9) MS(8, -1);
    if (!issued) {
      POLL_WAIT();
      poll_use();
      wait_ready(s + 2);
      stage_load(s + 1);
    }
    POLL_WAIT();
    poll_use();
  }
  {
    constexpr int s = NSTEP - 1;
    const char* sb = smem + (s & 1) * STAGE;
    __amdgpu_buffer_rsrc_t ro = __builtin_amdgcn_make_buffer_rsrc(P.out, 0, (unsigned)((size_t)MD * ND * 4), 0x00020000);
    const unsigned vo = ((unsigned)(TMS * tm + wm * 144 + fg * 4) * ND + TN * tn + wn * 64 + fj * 4) * 4u;
    __builtin_amdgcn_s_waitcnt(0xC07F);
    SCHED();
    __builtin_amdgcn_s_barrier();
    SCHED();
    POLL_ISSUE();
    frags0(sb);
    float sc = scale;
    auto store_m = [&](auto mc) {
      constexpr int m = decltype(mc)::value;
#pragma unroll
      for (int r = 0; r < 4; ++r) {
        f32x4 v = {acc[m][0][r] * sc, acc[m][1][r] * sc, acc[m][2][r] * sc, acc[m][3][r] * sc};
        __builtin_amdgcn_raw_buffer_store_b128(__builtin_bit_cast(u32x4, v), ro, vo + (unsigned)(m * 16 + r) * ND * 4u, 0, 16);
      }
      SCHED();
    };
#define ST(m) store_m(IC(m))
    MS(0, -1); MS(1, -1);
    POLL_WAIT();
    if ((unsigned)__builtin_amdgcn_readlane(pv, 3) == call) sc = __builtin_nanf("");
    SCHED();
    ST(0);
    MS(2, -1); ST(1);
    MS(3, -1); ST(2);
    MS(4, -1); ST(3);
    MS(5, -1); ST(4);
    MS(6, -1); ST(5);
    MS(7, -1); ST(6);
    if (full9) MS(8, -1);
    SCHED();
    asm volatile("s_nop 15\n\ts_nop 15" ::: "memory");
    SCHED();
    ST(7);
    if (full9) ST(8);
  }
}

extern "C" void kernel_launch(int rank, int nranks, ncclComm_t comm,
                              hipStream_t stream,
                              void* const* d_in, const int* in_sizes, int n_in,
                              void* d_out, int out_size,
                              const int* in_dtypes, int out_dtype,
                              void* const* d_out_all,
                              void* const* d_ws_all, size_t ws_bytes) {
  if (nranks != NR || ws_bytes < WS_NEED || (unsigned)rank >= (unsigned)NR || n_in < 4 ||
      in_sizes[0] != MD * KS || in_sizes[1] != KS * ND || out_size != MD * ND) {
    fprintf(stderr, "unexpected problem shape\n");
    return;
  }
  HIP_CHECK(hipFuncSetAttribute((const void*)gemm_ag_kernel, hipFuncAttributeMaxDynamicSharedMemorySize, LDS_BYTES));
  Params P;
  for (int p = 0; p < NR; ++p) P.wsr[p] = (char*)d_ws_all[(rank + p) & 7];
  P.pad = 0;
  P.x = (const unsigned char*)d_in[0];
  P.w = (const unsigned char*)d_in[1];
  P.sx = (const float*)d_in[2];
  P.sw = (const float*)d_in[3];
  P.out = (float*)d_out;
  P.rank = rank;
  hipLaunchKernelGGL(gemm_ag_kernel, dim3(GRID), dim3(512), LDS_BYTES, stream, P);
  HIP_CHECK(hipGetLastError());
}
```
